```python
import numpy as np

kernel_name = "gpu_ag_gemm_m4096_k4096_n32768_bf16_relu"


ABSMAX_RTOL = 0.008

COMPUTE_DTYPE = "bfloat16"
EPILOGUE = "relu"
IN_QUANT = None
OUT_QUANT = None
IN_DTYPES = {"x": "bfloat16", "w_mat": "bfloat16"}
WORLD_SIZE = 8

def _to_bf16(x):
    u = np.ascontiguousarray(x, dtype=np.float32).view(np.uint32)
    return ((u + 0x7FFF + ((u >> 16) & 1)) & 0xFFFF0000).view(np.float32)

def setup_inputs():

    x = _to_bf16(np.random.randn(4096, 4096).astype(np.float32))
    w_mat = _to_bf16((np.random.randn(4096, 32768) / np.sqrt(4096)).astype(np.float32))
    return {"x": x, "w_mat": w_mat}

def _epilogue(y):
    return np.maximum(y, 0.0).astype(np.float32)

def reference(x, w_mat):
    y = (x @ w_mat).astype(np.float32)
    return _epilogue(y)
```

```cpp
#include <hip/hip_runtime.h>
#include <rccl/rccl.h>
#include <cstdio>

#define LAS __attribute__((address_space(3)))
typedef short bf16x8 __attribute__((ext_vector_type(8)));
typedef short bf16x4 __attribute__((ext_vector_type(4)));
typedef float f32x4 __attribute__((ext_vector_type(4)));
typedef unsigned u32x4 __attribute__((ext_vector_type(4)));
typedef unsigned long long u64;

namespace {

constexpr int M_ALL = 4096, KDIM = 4096, NLOC = 4096, NRANK = 8, MSH = M_ALL / NRANK;
constexpr int BM = 256, BK = 64, HALF = 128, HTB = HALF * BK * 2, NXCD = 8, WGM = 8;
constexpr int STAGE_BYTES = 8 * HTB;
constexpr int LDS_BYTES = STAGE_BYTES + 64;
constexpr int NWG = (M_ALL / BM) * (NLOC / BM);
constexpr int CHUNK_COLS = 256, NCHUNK = KDIM / CHUNK_COLS;
constexpr size_t WS_CTR = 0;
constexpr size_t WS_TMO = 2048;
constexpr size_t WS_FLAGS = 4096;
constexpr size_t WS_GATHER = 1u << 20;
constexpr size_t GATHER_BYTES = (size_t)M_ALL * KDIM * 2;
constexpr size_t WS_NEED = WS_GATHER + 2 * GATHER_BYTES;
constexpr u64 DEADLINE_TICKS = 20000000ull;

__host__ __device__ __forceinline__ int lds_byte(int r, int c) {
  const int st = (r >> 4) * 2 + (c >> 5), rr = r & 15, cc = c & 31, ob = rr * 64 + cc * 2;
  return st * 1024 + (ob ^ (((ob >> 9) & 1) << 5));
}
__host__ __device__ __forceinline__ void stage_rc(int b, int& R, int& C) {
  const int st = b / 1024, sb = b % 1024, swz = sb ^ (((sb >> 9) & 1) << 5);
  R = (st >> 1) * 16 + swz / 64; C = (st & 1) * 32 + (swz % 64) / 2;
}
__host__ __device__ __forceinline__ int fswz(int row) { return ((row & 3) << 2) | ((row >> 2) & 3); }

__device__ __forceinline__ unsigned cvt_pk_bf16(float lo, float hi) {
  unsigned r; asm volatile("v_cvt_pk_bf16_f32 %0, %1, %2" : "=v"(r) : "v"(lo), "v"(hi)); return r;
}
__device__ __forceinline__ bf16x8 tr_frag(LAS unsigned char* lds, int a0, int a1) {
  bf16x4 lo = __builtin_amdgcn_ds_read_tr16_b64_v4i16((LAS bf16x4*)(lds + a0));
  bf16x4 hi = __builtin_amdgcn_ds_read_tr16_b64_v4i16((LAS bf16x4*)(lds + a1));
  bf16x8 r;
  r[0] = lo[0]; r[1] = lo[1]; r[2] = lo[2]; r[3] = lo[3];
  r[4] = hi[0]; r[5] = hi[1]; r[6] = hi[2]; r[7] = hi[3];
  return r;
}
__device__ __forceinline__ void store16_sys(u64 base, unsigned voff, u32x4 v) {
  asm volatile("global_store_dwordx4 %0, %1, %2 sc0 sc1\n\ts_nop 1" :: "v"(voff), "v"(v), "s"(base) : "memory");
}
__device__ __forceinline__ u64 lane_ptr(unsigned lo, unsigned hi, int p) {
  unsigned l = __builtin_amdgcn_readlane(lo, p), h = __builtin_amdgcn_readlane(hi, p);
  return ((u64)h << 32) | l;
}

__device__ __forceinline__ bool wait_flags(const u64* fp, int lane, unsigned seq, unsigned* tmo) {
  const u64 t0 = __builtin_amdgcn_s_memrealtime();
  for (;;) {
    u64 v = __hip_atomic_load(fp + lane, __ATOMIC_RELAXED, __HIP_MEMORY_SCOPE_SYSTEM);
    bool ok = ((unsigned)v - seq) < 0x80000000u && ((unsigned)(v >> 32) - seq) < 0x80000000u;
    if (__all(ok)) return true;
    unsigned tm = __hip_atomic_load(tmo, __ATOMIC_RELAXED, __HIP_MEMORY_SCOPE_AGENT);
    if (__builtin_amdgcn_readfirstlane(tm) == seq || __builtin_amdgcn_s_memrealtime() - t0 > DEADLINE_TICKS) {
      if (lane == 0) __hip_atomic_store(tmo, seq, __ATOMIC_RELAXED, __HIP_MEMORY_SCOPE_AGENT);
      return false;
    }
    __builtin_amdgcn_s_sleep(1);
  }
}

__global__ __launch_bounds__(512, 2)
void ag_gemm_fused(const unsigned short* __restrict__ X, const unsigned short* __restrict__ W, unsigned short* __restrict__ Cout,
                   void* const* __restrict__ ws_all, int rank) {
  extern __shared__ __attribute__((aligned(16))) unsigned char smem[];
  LAS unsigned char* lds = (LAS unsigned char*)smem;
  const int tid = threadIdx.x, wid = __builtin_amdgcn_readfirstlane(tid >> 6), lane = tid & 63, wr = wid >> 2, wc = wid & 3, fr = lane & 15, fq = lane >> 4;
  constexpr int K = KDIM, N = NLOC, nt = K / BK;
  unsigned char* myws = (unsigned char*)ws_all[rank];

  if (tid == 0) {
    unsigned old = __hip_atomic_fetch_add((unsigned*)(myws + WS_CTR + (blockIdx.x & 7) * 128), 1u, __ATOMIC_RELAXED, __HIP_MEMORY_SCOPE_AGENT);
    *(LAS unsigned*)(lds + STAGE_BYTES) = old / (NWG / 8) + 1;
  }
  u64 pw = (u64)ws_all[lane & 7];
  const unsigned pw_lo = (unsigned)pw, pw_hi = (unsigned)(pw >> 32);
  const unsigned xoff = (unsigned)(((2 * blockIdx.x + (lane >> 5)) * K + (lane & 31) * 8) * 2);
  u32x4 xd0 = *(const u32x4*)((const char*)X + xoff + wid * 512);
  u32x4 xd1 = *(const u32x4*)((const char*)X + xoff + (wid + 8) * 512);
  __syncthreads();
  const unsigned seq = __builtin_amdgcn_readfirstlane(*(LAS unsigned*)(lds + STAGE_BYTES));
  const size_t gbuf = WS_GATHER + (size_t)(seq & 1) * GATHER_BYTES;
  unsigned* tmo = (unsigned*)(myws + WS_TMO);

  {
    const unsigned doff = (unsigned)(rank * MSH * K * 2) + xoff;
#pragma unroll
    for (int i = 1; i < NRANK; ++i) { int p = (rank + i) & 7; u64 b = lane_ptr(pw_lo, pw_hi, p) + gbuf; store16_sys(b, doff + wid * 512, xd0); }
#pragma unroll
    for (int i = 1; i < NRANK; ++i) { int p = (rank + i) & 7; u64 b = lane_ptr(pw_lo, pw_hi, p) + gbuf; store16_sys(b, doff + (wid + 8) * 512, xd1); }
    asm volatile("s_waitcnt vmcnt(0)" ::: "memory");
    if (lane < 8 && lane != rank) {
      unsigned* f = (unsigned*)(pw + WS_FLAGS) + (rank * NCHUNK + wid) * NWG + blockIdx.x;
      __hip_atomic_store(f, seq, __ATOMIC_RELAXED, __HIP_MEMORY_SCOPE_SYSTEM);
      __hip_atomic_store(f + 8 * NWG, seq, __ATOMIC_RELAXED, __HIP_MEMORY_SCOPE_SYSTEM);
    }
  }

  int wgid = blockIdx.x;
  { const int xcd = wgid % NXCD, off = wgid / NXCD; wgid = xcd * (NWG / NXCD) + off; }
  constexpr int nN = N / BM, nig = WGM * nN;
  const int pm = (wgid / nig) * WGM + (wgid % nig) % WGM, pn = (wgid % nig) / WGM;
  const int src = pm >> 1;
  const bool remote = (src != rank);

  unsigned voffA[2], voffB[2];
#pragma unroll
  for (int i = 0; i < 2; ++i) {
    int R, C; stage_rc(tid * 16 + i * 8192, R, C);
    voffA[i] = (unsigned)(R * K + C) * 2u;
    int b = tid * 16 + i * 8192, row = b >> 8, chp = (b >> 4) & 15, ch = chp ^ fswz(row);
    voffB[i] = (unsigned)(row * N + ch * 8) * 2u;
  }
  constexpr size_t kstepA = (size_t)(BK * 2), hstepA = (size_t)HALF * K * 2, kstepB = (size_t)BK * N * 2, hstepB = (size_t)HALF * 2;
  const unsigned ldsw = (unsigned)wid * 1024u;
  const int aoff = lds_byte(wr * 64 + fr, fq * 8);
  int boff[2][2];
  { const int g = fq, q = fr >> 2, p = lane & 3;
#pragma unroll
    for (int n = 0; n < 2; ++n)
#pragma unroll
      for (int t = 0; t < 2; ++t) {
        int row = 8 * g + 4 * t + q; int chunk = 2 * (2 * wc + n) + (p >> 1);
        boff[n][t] = 256 * row + 16 * (chunk ^ fswz(row)) + 8 * (p & 1);
      }
  }
#define SA(b, h) (((b) * 2 + (h)) * HTB)
#define SB(b, h) ((4 + (b) * 2 + (h)) * HTB)
#define STAGE(bufoff, gbase, voff) do { _Pragma("unroll") for (int _i = 0; _i < 2; ++_i) \
    __builtin_amdgcn_global_load_lds((const unsigned*)((const char*)(gbase) + (voff)[_i]), (LAS unsigned*)(lds + (bufoff) + ldsw + _i * 8192), 16, 0, 0); } while (0)
#define LDA(dst, b, h) do { _Pragma("unroll") for (int m = 0; m < 4; ++m) _Pragma("unroll") for (int k = 0; k < 2; ++k) dst[m][k] = *(const LAS bf16x8*)(lds + SA(b, h) + aoff + m * 2048 + k * 1024); } while (0)
#define LDB(dst, b, h) do { _Pragma("unroll") for (int n = 0; n < 2; ++n) _Pragma("unroll") for (int k = 0; k < 2; ++k) dst[n][k] = tr_frag(lds + SB(b, h) + k * 8192, boff[n][0], boff[n][1]); } while (0)
#define MMA(ai, bj, At, Bt) do { __builtin_amdgcn_s_setprio(1); _Pragma("unroll") for (int m = 0; m < 4; ++m) _Pragma("unroll") for (int n = 0; n < 2; ++n) _Pragma("unroll") for (int k = 0; k < 2; ++k) \
    acc[ai][bj][m][n] = __builtin_amdgcn_mfma_f32_16x16x32_bf16(Bt[n][k], At[m][k], acc[ai][bj][m][n], 0, 0, 0); __builtin_amdgcn_s_setprio(0); } while (0)
#define WAIT_V(n) asm volatile("s_waitcnt vmcnt(" #n ")" ::: "memory")
#define WAIT_L(n) asm volatile("s_waitcnt lgkmcnt(" #n ")" ::: "memory")
#define BAR __builtin_amdgcn_s_barrier()
#define SCHED __builtin_amdgcn_sched_barrier(0)

  f32x4 acc[2][2][4][2];
#pragma unroll
  for (int a = 0; a < 2; ++a)
#pragma unroll
    for (int b = 0; b < 2; ++b)
#pragma unroll
      for (int m = 0; m < 4; ++m)
#pragma unroll
        for (int n = 0; n < 2; ++n) acc[a][b][m][n] = (f32x4){0.f, 0.f, 0.f, 0.f};
  bf16x8 At[4][2], B0[2][2], B1[2][2];
  const char* cA = (remote ? (const char*)(myws + gbuf) + (size_t)pm * 2 * hstepA : (const char*)X + (size_t)(pm & 1) * 2 * hstepA);
  const char* cB = (const char*)W + (size_t)pn * 2 * hstepB;
  const u64* myflags = (const u64*)((unsigned*)(myws + WS_FLAGS) + (size_t)src * NCHUNK * NWG + (pm & 1) * 128);

  if (remote && wid == 0) { wait_flags(myflags, lane, seq, tmo); }
  asm volatile("" ::: "memory");
  __syncthreads();

  STAGE(SB(0, 0), cB, voffB); STAGE(SA(0, 0), cA, voffA); STAGE(SB(0, 1), cB + hstepB, voffB); STAGE(SA(0, 1), cA + hstepA, voffA);
  if (wr == 1) BAR;
  WAIT_V(4); BAR;
  STAGE(SB(1, 0), cB + kstepB, voffB); STAGE(SA(1, 0), cA + kstepA, voffA); STAGE(SB(1, 1), cB + hstepB + kstepB, voffB);
  WAIT_V(6); BAR;
  for (int t = 0; t < nt; t += 2) {
    const bool last = (t == nt - 2);
    const char* a1 = cA + (size_t)(t + 1) * kstepA;
    const char* a2 = last ? cA : cA + (size_t)(t + 2) * kstepA; const char* b2 = last ? cB : cB + (size_t)(t + 2) * kstepB;
    const char* a3 = a2 + kstepA; const char* b3 = b2 + kstepB;
    if (remote && wid == 0 && (t & 2) && !last) wait_flags(myflags + (size_t)((t + 2) >> 2) * (NWG / 2), lane, seq, tmo);
    LDB(B0, 0, 0); SCHED; LDA(At, 0, 0); STAGE(SA(1, 1), a1 + hstepA, voffA);
    WAIT_L(8); BAR; WAIT_L(0); MMA(0, 0, At, B0); BAR; SCHED;
    LDB(B1, 0, 1); STAGE(SB(0, 0), b2, voffB);
    BAR; WAIT_L(0); MMA(0, 1, At, B1); BAR;
    LDA(At, 0, 1); STAGE(SA(0, 0), a2, voffA);
    BAR; WAIT_L(0); MMA(1, 0, At, B0); BAR; SCHED;
    STAGE(SB(0, 1), b2 + hstepB, voffB);
    WAIT_V(6); BAR; MMA(1, 1, At, B1); BAR;
    LDB(B0, 1, 0); SCHED; LDA(At, 1, 0); STAGE(SA(0, 1), a2 + hstepA, voffA);
    WAIT_L(8); BAR; WAIT_L(0); MMA(0, 0, At, B0); BAR; SCHED;
    LDB(B1, 1, 1); STAGE(SB(1, 0), b3, voffB);
    BAR; WAIT_L(0); MMA(0, 1, At, B1); BAR;
    LDA(At, 1, 1); STAGE(SA(1, 0), a3, voffA);
    BAR; WAIT_L(0); MMA(1, 0, At, B0); BAR; SCHED;
    STAGE(SB(1, 1), b3 + hstepB, voffB);
    WAIT_V(6); BAR; MMA(1, 1, At, B1); BAR;
  }
  WAIT_V(0);
  if (wr == 0) BAR;
  const bool bad = __builtin_amdgcn_readfirstlane(__hip_atomic_load(tmo, __ATOMIC_RELAXED, __HIP_MEMORY_SCOPE_AGENT)) == seq;
#pragma unroll
  for (int ai = 0; ai < 2; ++ai)
#pragma unroll
    for (int m = 0; m < 4; ++m) {
      unsigned short* rowp = Cout + (size_t)(pm * BM + ai * HALF + wr * 64 + m * 16 + fr) * N + pn * BM + wc * 32 + (fq & 1) * 16 + (fq >> 1) * 8;
#pragma unroll
      for (int bj = 0; bj < 2; ++bj) {
        f32x4 v0 = acc[ai][bj][m][0], v1 = acc[ai][bj][m][1];
        unsigned x0 = cvt_pk_bf16(fmaxf(v0[0], 0.f), fmaxf(v0[1], 0.f)), x1 = cvt_pk_bf16(fmaxf(v0[2], 0.f), fmaxf(v0[3], 0.f));
        unsigned y0 = cvt_pk_bf16(fmaxf(v1[0], 0.f), fmaxf(v1[1], 0.f)), y1 = cvt_pk_bf16(fmaxf(v1[2], 0.f), fmaxf(v1[3], 0.f));
        auto r0 = __builtin_amdgcn_permlane16_swap(x0, y0, false, false);
        auto r1 = __builtin_amdgcn_permlane16_swap(x1, y1, false, false);
        u32x4 w; w.x = r0[0]; w.y = r1[0]; w.z = r0[1]; w.w = r1[1];
        if (bad) w = (u32x4){0x7fc07fc0u, 0x7fc07fc0u, 0x7fc07fc0u, 0x7fc07fc0u};
        *(u32x4*)(rowp + bj * HALF) = w;
      }
    }
}

}

#define HIP_CHECK(expr) do { hipError_t _e = (expr); if (_e != hipSuccess) { \
    fprintf(stderr, "HIP error %s at %s:%d\n", hipGetErrorString(_e), __FILE__, __LINE__); \
    return; } } while (0)

extern "C" void kernel_launch(int rank, int nranks, ncclComm_t comm,
                              hipStream_t stream,
                              void* const* d_in, const int* in_sizes, int n_in,
                              void* d_out, int out_size,
                              const int* in_dtypes, int out_dtype,
                              void* const* d_out_all,
                              void* const* d_ws_all, size_t ws_bytes) {
  if (nranks != NRANK || ws_bytes < WS_NEED || n_in < 2 || in_sizes[0] != MSH * KDIM || in_sizes[1] != KDIM * NLOC || out_size != M_ALL * NLOC) {
    fprintf(stderr, "unexpected configuration\n"); return;
  }
  static bool init = false;
  if (!init) {
    HIP_CHECK(hipFuncSetAttribute((const void*)ag_gemm_fused, hipFuncAttributeMaxDynamicSharedMemorySize, LDS_BYTES));
    init = true;
  }
  ag_gemm_fused<<<NWG, 512, LDS_BYTES, stream>>>((const unsigned short*)d_in[0], (const unsigned short*)d_in[1],
                                                 (unsigned short*)d_out, d_ws_all, rank);
  HIP_CHECK(hipGetLastError());
}
```

```cpp
#include <hip/hip_runtime.h>
#include <rccl/rccl.h>
#include <cstdio>

#define LAS __attribute__((address_space(3)))
typedef short bf16x8 __attribute__((ext_vector_type(8)));
typedef short bf16x4 __attribute__((ext_vector_type(4)));
typedef float f32x4 __attribute__((ext_vector_type(4)));
typedef unsigned u32x4 __attribute__((ext_vector_type(4)));
typedef unsigned long long u64;

namespace {

constexpr int M_ALL = 4096, KDIM = 4096, NLOC = 4096, NRANK = 8, MSH = M_ALL / NRANK;
constexpr int BM = 256, BK = 64, HALF = 128, HTB = HALF * BK * 2, NXCD = 8, WGM = 8;
constexpr int STAGE_BYTES = 8 * HTB;
constexpr int LDS_BYTES = STAGE_BYTES + 64;
constexpr int NWG = (M_ALL / BM) * (NLOC / BM);
constexpr int CHUNK_COLS = 256, NCHUNK = KDIM / CHUNK_COLS;
constexpr size_t WS_CTR = 0;
constexpr size_t WS_TMO = 2048;
constexpr size_t WS_FLAGS = 4096;
constexpr size_t WS_GATHER = 1u << 20;
constexpr size_t GATHER_BYTES = (size_t)M_ALL * KDIM * 2;
constexpr size_t WS_NEED = WS_GATHER + 2 * GATHER_BYTES;
#define PRE_CHUNKS 2
constexpr u64 DEADLINE_TICKS = 20000000ull;

__host__ __device__ __forceinline__ int lds_byte(int r, int c) {
  const int st = (r >> 4) * 2 + (c >> 5), rr = r & 15, cc = c & 31, ob = rr * 64 + cc * 2;
  return st * 1024 + (ob ^ (((ob >> 9) & 1) << 5));
}
__host__ __device__ __forceinline__ void stage_rc(int b, int& R, int& C) {
  const int st = b / 1024, sb = b % 1024, swz = sb ^ (((sb >> 9) & 1) << 5);
  R = (st >> 1) * 16 + swz / 64; C = (st & 1) * 32 + (swz % 64) / 2;
}
__host__ __device__ __forceinline__ int fswz(int row) { return ((row & 3) << 2) | ((row >> 2) & 3); }

__device__ __forceinline__ unsigned cvt_pk_bf16(float lo, float hi) {
  unsigned r; asm volatile("v_cvt_pk_bf16_f32 %0, %1, %2" : "=v"(r) : "v"(lo), "v"(hi)); return r;
}
__device__ __forceinline__ bf16x8 tr_frag(LAS unsigned char* lds, int a0, int a1) {
  bf16x4 lo = __builtin_amdgcn_ds_read_tr16_b64_v4i16((LAS bf16x4*)(lds + a0));
  bf16x4 hi = __builtin_amdgcn_ds_read_tr16_b64_v4i16((LAS bf16x4*)(lds + a1));
  bf16x8 r;
  r[0] = lo[0]; r[1] = lo[1]; r[2] = lo[2]; r[3] = lo[3];
  r[4] = hi[0]; r[5] = hi[1]; r[6] = hi[2]; r[7] = hi[3];
  return r;
}
__device__ __forceinline__ void store16_sys(u64 base, unsigned voff, u32x4 v) {
  asm volatile("global_store_dwordx4 %0, %1, %2 sc0 sc1\n\ts_nop 1" :: "v"(voff), "v"(v), "s"(base) : "memory");
}
__device__ __forceinline__ u64 lane_ptr(unsigned lo, unsigned hi, int p) {
  unsigned l = __builtin_amdgcn_readlane(lo, p), h = __builtin_amdgcn_readlane(hi, p);
  return ((u64)h << 32) | l;
}

__device__ __forceinline__ bool wait_flags(const u64* fp, int lane, unsigned seq, unsigned* tmo) {
  const u64 t0 = __builtin_amdgcn_s_memrealtime();
  for (;;) {
    u64 v = __hip_atomic_load(fp + lane, __ATOMIC_RELAXED, __HIP_MEMORY_SCOPE_SYSTEM);
    bool ok = ((unsigned)v - seq) < 0x80000000u && ((unsigned)(v >> 32) - seq) < 0x80000000u;
    if (__all(ok)) return true;
    unsigned tm = __hip_atomic_load(tmo, __ATOMIC_RELAXED, __HIP_MEMORY_SCOPE_AGENT);
    if (__builtin_amdgcn_readfirstlane(tm) == seq || __builtin_amdgcn_s_memrealtime() - t0 > DEADLINE_TICKS) {
      if (lane == 0) __hip_atomic_store(tmo, seq, __ATOMIC_RELAXED, __HIP_MEMORY_SCOPE_AGENT);
      return false;
    }
    __builtin_amdgcn_s_sleep(1);
  }
}

__global__ __launch_bounds__(512, 2)
void ag_gemm_fused(const unsigned short* __restrict__ X, const unsigned short* __restrict__ W, unsigned short* __restrict__ Cout,
                   void* const* __restrict__ ws_all, int rank) {
  extern __shared__ __attribute__((aligned(16))) unsigned char smem[];
  LAS unsigned char* lds = (LAS unsigned char*)smem;
  const int tid = threadIdx.x, wid = __builtin_amdgcn_readfirstlane(tid >> 6), lane = tid & 63, wr = wid >> 2, wc = wid & 3, fr = lane & 15, fq = lane >> 4;
  constexpr int K = KDIM, N = NLOC, nt = K / BK;
  unsigned char* myws = (unsigned char*)ws_all[rank];

  if (tid == 0) {
    unsigned old = __hip_atomic_fetch_add((unsigned*)(myws + WS_CTR + (blockIdx.x & 7) * 128), 1u, __ATOMIC_RELAXED, __HIP_MEMORY_SCOPE_AGENT);
    *(LAS unsigned*)(lds + STAGE_BYTES) = old / (NWG / 8) + 1;
  }
  u64 pw = (u64)ws_all[lane & 7];
  const unsigned pw_lo = (unsigned)pw, pw_hi = (unsigned)(pw >> 32);
  const unsigned xoff = (unsigned)(((2 * blockIdx.x + (lane >> 5)) * K + (lane & 31) * 8) * 2);
  u32x4 xd0 = *(const u32x4*)((const char*)X + xoff + wid * 512);
  u32x4 xd1 = *(const u32x4*)((const char*)X + xoff + (wid + 8) * 512);
  __syncthreads();
  const unsigned seq = __builtin_amdgcn_readfirstlane(*(LAS unsigned*)(lds + STAGE_BYTES));
  const size_t gbuf = WS_GATHER + (size_t)(seq & 1) * GATHER_BYTES;
  unsigned* tmo = (unsigned*)(myws + WS_TMO);

  const unsigned doff = (unsigned)(rank * MSH * K * 2) + xoff;
  int pc = 0, pp = 0;
  int cflag = 0;
#define PUSH_SLOT() do { if (pc < NCHUNK) { if (wid == (pc & 7)) { \
      u64 b_ = lane_ptr(pw_lo, pw_hi, (rank + 1 + pp) & 7) + gbuf; \
      store16_sys(b_, doff + pc * 512, pc < 8 ? xd0 : xd1); } \
      if (++pp == 7) { pp = 0; ++pc; } } } while (0)
#define SEND_FLAG(c) do { if (wid == ((c) & 7) && lane < 8 && lane != rank) { \
      unsigned* f_ = (unsigned*)(pw + WS_FLAGS) + (rank * NCHUNK + (c)) * NWG + blockIdx.x; \
      __hip_atomic_store(f_, seq, __ATOMIC_RELAXED, __HIP_MEMORY_SCOPE_SYSTEM); } } while (0)
#pragma unroll
  for (int i = 0; i < 7; ++i) PUSH_SLOT();
  if (wid == 0) { asm volatile("s_waitcnt vmcnt(0)" ::: "memory"); }
  SEND_FLAG(0); cflag = 1;
  for (int i = 0; i < 7 * (PRE_CHUNKS - 1); ++i) PUSH_SLOT();

  int wgid = blockIdx.x;
  { const int xcd = wgid % NXCD, off = wgid / NXCD; wgid = xcd * (NWG / NXCD) + off; }
  constexpr int nN = N / BM, nig = WGM * nN;
  const int pm = (wgid / nig) * WGM + (wgid % nig) % WGM, pn = (wgid % nig) / WGM;
  const int src = pm >> 1;
  const bool remote = (src != rank);

  unsigned voffA[2], voffB[2];
#pragma unroll
  for (int i = 0; i < 2; ++i) {
    int R, C; stage_rc(tid * 16 + i * 8192, R, C);
    voffA[i] = (unsigned)(R * K + C) * 2u;
    int b = tid * 16 + i * 8192, row = b >> 8, chp = (b >> 4) & 15, ch = chp ^ fswz(row);
    voffB[i] = (unsigned)(row * N + ch * 8) * 2u;
  }
  constexpr size_t kstepA = (size_t)(BK * 2), hstepA = (size_t)HALF * K * 2, kstepB = (size_t)BK * N * 2, hstepB = (size_t)HALF * 2;
  const unsigned ldsw = (unsigned)wid * 1024u;
  const int aoff = lds_byte(wr * 64 + fr, fq * 8);
  int boff[2][2];
  { const int g = fq, q = fr >> 2, p = lane & 3;
#pragma unroll
    for (int n = 0; n < 2; ++n)
#pragma unroll
      for (int t = 0; t < 2; ++t) {
        int row = 8 * g + 4 * t + q; int chunk = 2 * (2 * wc + n) + (p >> 1);
        boff[n][t] = 256 * row + 16 * (chunk ^ fswz(row)) + 8 * (p & 1);
      }
  }
#define SA(b, h) (((b) * 2 + (h)) * HTB)
#define SB(b, h) ((4 + (b) * 2 + (h)) * HTB)
#define STAGE(bufoff, gbase, voff) do { _Pragma("unroll") for (int _i = 0; _i < 2; ++_i) \
    __builtin_amdgcn_global_load_lds((const unsigned*)((const char*)(gbase) + (voff)[_i]), (LAS unsigned*)(lds + (bufoff) + ldsw + _i * 8192), 16, 0, 0); } while (0)
#define LDA(dst, b, h) do { _Pragma("unroll") for (int m = 0; m < 4; ++m) _Pragma("unroll") for (int k = 0; k < 2; ++k) dst[m][k] = *(const LAS bf16x8*)(lds + SA(b, h) + aoff + m * 2048 + k * 1024); } while (0)
#define LDB(dst, b, h) do { _Pragma("unroll") for (int n = 0; n < 2; ++n) _Pragma("unroll") for (int k = 0; k < 2; ++k) dst[n][k] = tr_frag(lds + SB(b, h) + k * 8192, boff[n][0], boff[n][1]); } while (0)
#define MMA(ai, bj, At, Bt) do { __builtin_amdgcn_s_setprio(1); _Pragma("unroll") for (int m = 0; m < 4; ++m) _Pragma("unroll") for (int n = 0; n < 2; ++n) _Pragma("unroll") for (int k = 0; k < 2; ++k) \
    acc[ai][bj][m][n] = __builtin_amdgcn_mfma_f32_16x16x32_bf16(Bt[n][k], At[m][k], acc[ai][bj][m][n], 0, 0, 0); __builtin_amdgcn_s_setprio(0); } while (0)
#define WAIT_V(n) asm volatile("s_waitcnt vmcnt(" #n ")" ::: "memory")
#define WAIT_L(n) asm volatile("s_waitcnt lgkmcnt(" #n ")" ::: "memory")
#define BAR __builtin_amdgcn_s_barrier()
#define SCHED __builtin_amdgcn_sched_barrier(0)

  f32x4 acc[2][2][4][2];
#pragma unroll
  for (int a = 0; a < 2; ++a)
#pragma unroll
    for (int b = 0; b < 2; ++b)
#pragma unroll
      for (int m = 0; m < 4; ++m)
#pragma unroll
        for (int n = 0; n < 2; ++n) acc[a][b][m][n] = (f32x4){0.f, 0.f, 0.f, 0.f};
  bf16x8 At[4][2], B0[2][2], B1[2][2];
  const char* cA = (remote ? (const char*)(myws + gbuf) + (size_t)pm * 2 * hstepA : (const char*)X + (size_t)(pm & 1) * 2 * hstepA);
  const char* cB = (const char*)W + (size_t)pn * 2 * hstepB;
  const u64* myflags = (const u64*)((unsigned*)(myws + WS_FLAGS) + (size_t)src * NCHUNK * NWG + (pm & 1) * 128);

  if (remote && wid == 0) { wait_flags(myflags, lane, seq, tmo); }
  asm volatile("" ::: "memory");
  __syncthreads();

  STAGE(SB(0, 0), cB, voffB); STAGE(SA(0, 0), cA, voffA); STAGE(SB(0, 1), cB + hstepB, voffB); STAGE(SA(0, 1), cA + hstepA, voffA);
  if (wr == 1) BAR;
  WAIT_V(4); BAR;
  STAGE(SB(1, 0), cB + kstepB, voffB); STAGE(SA(1, 0), cA + kstepA, voffA); STAGE(SB(1, 1), cB + hstepB + kstepB, voffB);
  WAIT_V(6); BAR;
  for (int t = 0; t < nt; t += 2) {
    const bool last = (t == nt - 2);
    const char* a1 = cA + (size_t)(t + 1) * kstepA;
    const char* a2 = last ? cA : cA + (size_t)(t + 2) * kstepA; const char* b2 = last ? cB : cB + (size_t)(t + 2) * kstepB;
    const char* a3 = a2 + kstepA; const char* b3 = b2 + kstepB;
    const int cdone = pc;
    if (remote && wid == 0 && (t & 2) && !last) wait_flags(myflags + (size_t)((t + 2) >> 2) * (NWG / 2), lane, seq, tmo);
    LDB(B0, 0, 0); SCHED; LDA(At, 0, 0); STAGE(SA(1, 1), a1 + hstepA, voffA);
    WAIT_L(8); BAR; WAIT_L(0); MMA(0, 0, At, B0); BAR; SCHED;
    LDB(B1, 0, 1); STAGE(SB(0, 0), b2, voffB); PUSH_SLOT();
    BAR; WAIT_L(0); MMA(0, 1, At, B1); BAR;
    LDA(At, 0, 1); STAGE(SA(0, 0), a2, voffA);
    BAR; WAIT_L(0); MMA(1, 0, At, B0); BAR; SCHED;
    STAGE(SB(0, 1), b2 + hstepB, voffB); PUSH_SLOT();
    WAIT_V(6); BAR; MMA(1, 1, At, B1); BAR;
    LDB(B0, 1, 0); SCHED; LDA(At, 1, 0); STAGE(SA(0, 1), a2 + hstepA, voffA);
    WAIT_L(8); BAR; WAIT_L(0); MMA(0, 0, At, B0); BAR; SCHED;
    LDB(B1, 1, 1); STAGE(SB(1, 0), b3, voffB); PUSH_SLOT();
    BAR; WAIT_L(0); MMA(0, 1, At, B1); BAR;
    LDA(At, 1, 1); STAGE(SA(1, 0), a3, voffA);
    BAR; WAIT_L(0); MMA(1, 0, At, B0); BAR; SCHED;
    STAGE(SB(1, 1), b3 + hstepB, voffB);
    WAIT_V(6);
    if (cflag < cdone) { SEND_FLAG(cflag); ++cflag; }
    PUSH_SLOT();
    BAR; MMA(1, 1, At, B1); BAR;
  }
  WAIT_V(0);
  if (wr == 0) BAR;
  const bool bad = __builtin_amdgcn_readfirstlane(__hip_atomic_load(tmo, __ATOMIC_RELAXED, __HIP_MEMORY_SCOPE_AGENT)) == seq;
#pragma unroll
  for (int ai = 0; ai < 2; ++ai)
#pragma unroll
    for (int m = 0; m < 4; ++m) {
      unsigned short* rowp = Cout + (size_t)(pm * BM + ai * HALF + wr * 64 + m * 16 + fr) * N + pn * BM + wc * 32 + (fq & 1) * 16 + (fq >> 1) * 8;
#pragma unroll
      for (int bj = 0; bj < 2; ++bj) {
        f32x4 v0 = acc[ai][bj][m][0], v1 = acc[ai][bj][m][1];
        unsigned x0 = cvt_pk_bf16(fmaxf(v0[0], 0.f), fmaxf(v0[1], 0.f)), x1 = cvt_pk_bf16(fmaxf(v0[2], 0.f), fmaxf(v0[3], 0.f));
        unsigned y0 = cvt_pk_bf16(fmaxf(v1[0], 0.f), fmaxf(v1[1], 0.f)), y1 = cvt_pk_bf16(fmaxf(v1[2], 0.f), fmaxf(v1[3], 0.f));
        auto r0 = __builtin_amdgcn_permlane16_swap(x0, y0, false, false);
        auto r1 = __builtin_amdgcn_permlane16_swap(x1, y1, false, false);
        u32x4 w; w.x = r0[0]; w.y = r1[0]; w.z = r0[1]; w.w = r1[1];
        if (bad) w = (u32x4){0x7fc07fc0u, 0x7fc07fc0u, 0x7fc07fc0u, 0x7fc07fc0u};
        *(u32x4*)(rowp + bj * HALF) = w;
      }
    }
}

}

#define HIP_CHECK(expr) do { hipError_t _e = (expr); if (_e != hipSuccess) { \
    fprintf(stderr, "HIP error %s at %s:%d\n", hipGetErrorString(_e), __FILE__, __LINE__); \
    return; } } while (0)

extern "C" void kernel_launch(int rank, int nranks, ncclComm_t comm,
                              hipStream_t stream,
                              void* const* d_in, const int* in_sizes, int n_in,
                              void* d_out, int out_size,
                              const int* in_dtypes, int out_dtype,
                              void* const* d_out_all,
                              void* const* d_ws_all, size_t ws_bytes) {
  if (nranks != NRANK || ws_bytes < WS_NEED || n_in < 2 || in_sizes[0] != MSH * KDIM || in_sizes[1] != KDIM * NLOC || out_size != M_ALL * NLOC) {
    fprintf(stderr, "unexpected configuration\n"); return;
  }
  static bool init = false;
  if (!init) {
    HIP_CHECK(hipFuncSetAttribute((const void*)ag_gemm_fused, hipFuncAttributeMaxDynamicSharedMemorySize, LDS_BYTES));
    init = true;
  }
  ag_gemm_fused<<<NWG, 512, LDS_BYTES, stream>>>((const unsigned short*)d_in[0], (const unsigned short*)d_in[1],
                                                 (unsigned short*)d_out, d_ws_all, rank);
  HIP_CHECK(hipGetLastError());
}
```

```cpp
#include <hip/hip_runtime.h>
#include <rccl/rccl.h>
#include <cstdio>

#define LAS __attribute__((address_space(3)))
typedef short bf16x8 __attribute__((ext_vector_type(8)));
typedef short bf16x4 __attribute__((ext_vector_type(4)));
typedef float f32x4 __attribute__((ext_vector_type(4)));
typedef unsigned u32x4 __attribute__((ext_vector_type(4)));
typedef unsigned long long u64;

namespace {

constexpr int M_ALL = 4096, KDIM = 4096, NLOC = 4096, NRANK = 8, MSH = M_ALL / NRANK;
constexpr int BM = 256, BK = 64, HALF = 128, HTB = HALF * BK * 2, NXCD = 8, WGM = 8;
constexpr int STAGE_BYTES = 8 * HTB;
constexpr int LDS_BYTES = STAGE_BYTES + 64;
constexpr int NWG = (M_ALL / BM) * (NLOC / BM);
constexpr int CHUNK_COLS = 256, NCHUNK = KDIM / CHUNK_COLS;
constexpr size_t WS_CTR = 0;
constexpr size_t WS_TMO = 2048;
constexpr size_t WS_FLAGS = 4096;
constexpr size_t WS_STAMP = 512u << 10;
constexpr size_t WS_GATHER = 1u << 20;
constexpr size_t GATHER_BYTES = (size_t)M_ALL * KDIM * 2;
constexpr size_t WS_NEED = WS_GATHER + 2 * GATHER_BYTES;
#define PRE_CHUNKS 2
constexpr u64 DEADLINE_TICKS = 20000000ull;

__host__ __device__ __forceinline__ int lds_byte(int r, int c) {
  const int st = (r >> 4) * 2 + (c >> 5), rr = r & 15, cc = c & 31, ob = rr * 64 + cc * 2;
  return st * 1024 + (ob ^ (((ob >> 9) & 1) << 5));
}
__host__ __device__ __forceinline__ void stage_rc(int b, int& R, int& C) {
  const int st = b / 1024, sb = b % 1024, swz = sb ^ (((sb >> 9) & 1) << 5);
  R = (st >> 1) * 16 + swz / 64; C = (st & 1) * 32 + (swz % 64) / 2;
}
__host__ __device__ __forceinline__ int fswz(int row) { return ((row & 3) << 2) | ((row >> 2) & 3); }

__device__ __forceinline__ unsigned cvt_pk_bf16(float lo, float hi) {
  unsigned r; asm volatile("v_cvt_pk_bf16_f32 %0, %1, %2" : "=v"(r) : "v"(lo), "v"(hi)); return r;
}
__device__ __forceinline__ bf16x8 tr_frag(LAS unsigned char* lds, int a0, int a1) {
  bf16x4 lo = __builtin_amdgcn_ds_read_tr16_b64_v4i16((LAS bf16x4*)(lds + a0));
  bf16x4 hi = __builtin_amdgcn_ds_read_tr16_b64_v4i16((LAS bf16x4*)(lds + a1));
  bf16x8 r;
  r[0] = lo[0]; r[1] = lo[1]; r[2] = lo[2]; r[3] = lo[3];
  r[4] = hi[0]; r[5] = hi[1]; r[6] = hi[2]; r[7] = hi[3];
  return r;
}
__device__ __forceinline__ void store16_sys(u64 base, unsigned voff, u32x4 v) {
  asm volatile("global_store_dwordx4 %0, %1, %2 sc0 sc1\n\ts_nop 1" :: "v"(voff), "v"(v), "s"(base) : "memory");
}
__device__ __forceinline__ u64 lane_ptr(unsigned lo, unsigned hi, int p) {
  unsigned l = __builtin_amdgcn_readlane(lo, p), h = __builtin_amdgcn_readlane(hi, p);
  return ((u64)h << 32) | l;
}

__device__ __forceinline__ bool wait_flags(const u64* fp, int lane, unsigned seq, unsigned* tmo) {
  const u64 t0 = __builtin_amdgcn_s_memrealtime();
  for (;;) {
    u64 v = __hip_atomic_load(fp + lane, __ATOMIC_RELAXED, __HIP_MEMORY_SCOPE_SYSTEM);
    bool ok = ((unsigned)v - seq) < 0x80000000u && ((unsigned)(v >> 32) - seq) < 0x80000000u;
    if (__all(ok)) return true;
    unsigned tm = __hip_atomic_load(tmo, __ATOMIC_RELAXED, __HIP_MEMORY_SCOPE_AGENT);
    if (__builtin_amdgcn_readfirstlane(tm) == seq || __builtin_amdgcn_s_memrealtime() - t0 > DEADLINE_TICKS) {
      if (lane == 0) __hip_atomic_store(tmo, seq, __ATOMIC_RELAXED, __HIP_MEMORY_SCOPE_AGENT);
      return false;
    }
    __builtin_amdgcn_s_sleep(1);
  }
}

__global__ __launch_bounds__(512, 2)
void ag_gemm_fused(const unsigned short* __restrict__ X, const unsigned short* __restrict__ W, unsigned short* __restrict__ Cout,
                   void* const* __restrict__ ws_all, int rank) {
  extern __shared__ __attribute__((aligned(16))) unsigned char smem[];
  LAS unsigned char* lds = (LAS unsigned char*)smem;
  const int tid = threadIdx.x, wid = __builtin_amdgcn_readfirstlane(tid >> 6), lane = tid & 63, wr = wid >> 2, wc = wid & 3, fr = lane & 15, fq = lane >> 4;
  constexpr int K = KDIM, N = NLOC, nt = K / BK;
  unsigned char* myws = (unsigned char*)ws_all[rank];
#define STAMP() do {} while (0)
  STAMP();

  if (tid == 0) {
    unsigned old = __hip_atomic_fetch_add((unsigned*)(myws + WS_CTR + (blockIdx.x & 7) * 128), 1u, __ATOMIC_RELAXED, __HIP_MEMORY_SCOPE_AGENT);
    *(LAS unsigned*)(lds + STAGE_BYTES) = old / (NWG / 8) + 1;
  }
  u64 pw = (u64)ws_all[lane & 7];
  const unsigned pw_lo = (unsigned)pw, pw_hi = (unsigned)(pw >> 32);
  const unsigned xoff = (unsigned)(((2 * blockIdx.x + (lane >> 5)) * K + (lane & 31) * 8) * 2);
  u32x4 xd0 = *(const u32x4*)((const char*)X + xoff + wid * 512);
  u32x4 xd1 = *(const u32x4*)((const char*)X + xoff + (wid + 8) * 512);
  __syncthreads();
  const unsigned seq = __builtin_amdgcn_readfirstlane(*(LAS unsigned*)(lds + STAGE_BYTES));
  STAMP();
  const size_t gbuf = WS_GATHER + (size_t)(seq & 1) * GATHER_BYTES;
  unsigned* tmo = (unsigned*)(myws + WS_TMO);

  const unsigned doff = (unsigned)(rank * MSH * K * 2) + xoff;
  int pc = 0, pp = 0;
  const int rot = blockIdx.x % 7;
  int cflag = 0;
  int ns = 0;
#define WAIT_VS(k) do { if ((k) == 0) WAIT_V(6); else if ((k) == 1) WAIT_V(7); else WAIT_V(8); } while (0)
#define DBG_PEER(x) (x)
#define DBG_LANES true
#define PUSH_ON 1
#define PUSH_SLOT() do { if (PUSH_ON && pc < NCHUNK) { if (wid == (pc & 7)) { \
      int q_ = pp + rot; q_ = q_ >= 7 ? q_ - 7 : q_; \
      u64 b_ = lane_ptr(pw_lo, pw_hi, DBG_PEER((rank + 1 + q_) & 7)) + gbuf; \
      if (DBG_LANES) store16_sys(b_, doff + pc * 512, pc < 8 ? xd0 : xd1); ++ns; } \
      if (++pp == 7) { pp = 0; ++pc; } } } while (0)
#define SEND_FLAG(c) do { if (wid == ((c) & 7) && lane < 8 && lane != rank) { \
      unsigned* f_ = (unsigned*)(pw + WS_FLAGS) + (rank * NCHUNK + (c)) * NWG + blockIdx.x; \
      __hip_atomic_store(f_, seq, __ATOMIC_RELAXED, __HIP_MEMORY_SCOPE_SYSTEM); } } while (0)
#pragma unroll
  for (int i = 0; i < 7; ++i) PUSH_SLOT();
  if (wid == 0) { asm volatile("s_waitcnt vmcnt(0)" ::: "memory"); }
  SEND_FLAG(0); cflag = 1;
  for (int i = 0; i < 7 * (PRE_CHUNKS - 1); ++i) PUSH_SLOT();

  int wgid = blockIdx.x;
  { const int xcd = wgid % NXCD, off = wgid / NXCD; wgid = xcd * (NWG / NXCD) + off; }
  constexpr int nN = N / BM, nig = WGM * nN;
  const int pm = (wgid / nig) * WGM + (wgid % nig) % WGM, pn = (wgid % nig) / WGM;
  const int src = pm >> 1;
  const bool remote = (src != rank);

  unsigned voffA[2], voffB[2];
#pragma unroll
  for (int i = 0; i < 2; ++i) {
    int R, C; stage_rc(tid * 16 + i * 8192, R, C);
    voffA[i] = (unsigned)(R * K + C) * 2u;
    int b = tid * 16 + i * 8192, row = b >> 8, chp = (b >> 4) & 15, ch = chp ^ fswz(row);
    voffB[i] = (unsigned)(row * N + ch * 8) * 2u;
  }
  constexpr size_t kstepA = (size_t)(BK * 2), hstepA = (size_t)HALF * K * 2, kstepB = (size_t)BK * N * 2, hstepB = (size_t)HALF * 2;
  const unsigned ldsw = (unsigned)wid * 1024u;
  const int aoff = lds_byte(wr * 64 + fr, fq * 8);
  int boff[2][2];
  { const int g = fq, q = fr >> 2, p = lane & 3;
#pragma unroll
    for (int n = 0; n < 2; ++n)
#pragma unroll
      for (int t = 0; t < 2; ++t) {
        int row = 8 * g + 4 * t + q; int chunk = 2 * (2 * wc + n) + (p >> 1);
        boff[n][t] = 256 * row + 16 * (chunk ^ fswz(row)) + 8 * (p & 1);
      }
  }
#define SA(b, h) (((b) * 2 + (h)) * HTB)
#define SB(b, h) ((4 + (b) * 2 + (h)) * HTB)
#define STAGE(bufoff, gbase, voff) do { _Pragma("unroll") for (int _i = 0; _i < 2; ++_i) \
    __builtin_amdgcn_global_load_lds((const unsigned*)((const char*)(gbase) + (voff)[_i]), (LAS unsigned*)(lds + (bufoff) + ldsw + _i * 8192), 16, 0, 0); } while (0)
#define LDA(dst, b, h) do { _Pragma("unroll") for (int m = 0; m < 4; ++m) _Pragma("unroll") for (int k = 0; k < 2; ++k) dst[m][k] = *(const LAS bf16x8*)(lds + SA(b, h) + aoff + m * 2048 + k * 1024); } while (0)
#define LDB(dst, b, h) do { _Pragma("unroll") for (int n = 0; n < 2; ++n) _Pragma("unroll") for (int k = 0; k < 2; ++k) dst[n][k] = tr_frag(lds + SB(b, h) + k * 8192, boff[n][0], boff[n][1]); } while (0)
#define MMA(ai, bj, At, Bt) do { __builtin_amdgcn_s_setprio(1); _Pragma("unroll") for (int m = 0; m < 4; ++m) _Pragma("unroll") for (int n = 0; n < 2; ++n) _Pragma("unroll") for (int k = 0; k < 2; ++k) \
    acc[ai][bj][m][n] = __builtin_amdgcn_mfma_f32_16x16x32_bf16(Bt[n][k], At[m][k], acc[ai][bj][m][n], 0, 0, 0); __builtin_amdgcn_s_setprio(0); } while (0)
#define MMA_P(ai, bj, At, Bt) do { __builtin_amdgcn_s_setprio(1); _Pragma("unroll") for (int m = 0; m < 4; ++m) { _Pragma("unroll") for (int n = 0; n < 2; ++n) _Pragma("unroll") for (int k = 0; k < 2; ++k) \
    acc[ai][bj][m][n] = __builtin_amdgcn_mfma_f32_16x16x32_bf16(Bt[n][k], At[m][k], acc[ai][bj][m][n], 0, 0, 0); if (m == 0 || m == 2) PUSH_SLOT(); } __builtin_amdgcn_s_setprio(0); } while (0)
#define WAIT_V(n) asm volatile("s_waitcnt vmcnt(" #n ")" ::: "memory")
#define WAIT_L(n) asm volatile("s_waitcnt lgkmcnt(" #n ")" ::: "memory")
#define BAR __builtin_amdgcn_s_barrier()
#define SCHED __builtin_amdgcn_sched_barrier(0)

  f32x4 acc[2][2][4][2];
#pragma unroll
  for (int a = 0; a < 2; ++a)
#pragma unroll
    for (int b = 0; b < 2; ++b)
#pragma unroll
      for (int m = 0; m < 4; ++m)
#pragma unroll
        for (int n = 0; n < 2; ++n) acc[a][b][m][n] = (f32x4){0.f, 0.f, 0.f, 0.f};
  bf16x8 At[4][2], B0[2][2], B1[2][2];
  const char* cA = (remote ? (const char*)(myws + gbuf) + (size_t)pm * 2 * hstepA : (const char*)X + (size_t)(pm & 1) * 2 * hstepA);
  const char* cB = (const char*)W + (size_t)pn * 2 * hstepB;
  const u64* myflags = (const u64*)((unsigned*)(myws + WS_FLAGS) + (size_t)src * NCHUNK * NWG + (pm & 1) * 128);

  if (remote && wid == 0) { wait_flags(myflags, lane, seq, tmo); }
  asm volatile("" ::: "memory");
  __syncthreads();

  STAMP();
  STAGE(SB(0, 0), cB, voffB); STAGE(SA(0, 0), cA, voffA); STAGE(SB(0, 1), cB + hstepB, voffB); STAGE(SA(0, 1), cA + hstepA, voffA);
  if (wr == 1) BAR;
  WAIT_V(4); BAR;
  STAGE(SB(1, 0), cB + kstepB, voffB); STAGE(SA(1, 0), cA + kstepA, voffA); STAGE(SB(1, 1), cB + hstepB + kstepB, voffB);
  WAIT_V(6); BAR;
  for (int t = 0; t < nt; t += 2) {
    const bool last = (t == nt - 2);
    STAMP();
    const char* a1 = cA + (size_t)(t + 1) * kstepA;
    const char* a2 = last ? cA : cA + (size_t)(t + 2) * kstepA; const char* b2 = last ? cB : cB + (size_t)(t + 2) * kstepB;
    const char* a3 = a2 + kstepA; const char* b3 = b2 + kstepB;
    const int cdone = pc;
    if (remote && wid == 0 && (t & 2) && !last) wait_flags(myflags + (size_t)((t + 2) >> 2) * (NWG / 2), lane, seq, tmo);
    LDB(B0, 0, 0); SCHED; LDA(At, 0, 0); STAGE(SA(1, 1), a1 + hstepA, voffA); ns = 0; PUSH_SLOT(); PUSH_SLOT();
    WAIT_L(8); BAR; WAIT_L(0); MMA(0, 0, At, B0); BAR; SCHED;
    LDB(B1, 0, 1); STAGE(SB(0, 0), b2, voffB);
    BAR; WAIT_L(0); MMA(0, 1, At, B1); BAR;
    LDA(At, 0, 1); STAGE(SA(0, 0), a2, voffA);
    BAR; WAIT_L(0); MMA(1, 0, At, B0); BAR; SCHED;
    STAGE(SB(0, 1), b2 + hstepB, voffB);
    WAIT_VS(ns); BAR; MMA(1, 1, At, B1); BAR;
    LDB(B0, 1, 0); SCHED; LDA(At, 1, 0); STAGE(SA(0, 1), a2 + hstepA, voffA); ns = 0; PUSH_SLOT(); PUSH_SLOT();
    WAIT_L(8); BAR; WAIT_L(0); MMA(0, 0, At, B0); BAR; SCHED;
    LDB(B1, 1, 1); STAGE(SB(1, 0), b3, voffB);
    BAR; WAIT_L(0); MMA(0, 1, At, B1); BAR;
    LDA(At, 1, 1); STAGE(SA(1, 0), a3, voffA);
    BAR; WAIT_L(0); MMA(1, 0, At, B0); BAR; SCHED;
    STAGE(SB(1, 1), b3 + hstepB, voffB);
    WAIT_VS(ns);
    if (cflag < cdone) { SEND_FLAG(cflag); ++cflag; }
    BAR; MMA(1, 1, At, B1); BAR;
  }
  WAIT_V(0);
  if (wr == 0) BAR;
  STAMP();
  const bool bad = __builtin_amdgcn_readfirstlane(__hip_atomic_load(tmo, __ATOMIC_RELAXED, __HIP_MEMORY_SCOPE_AGENT)) == seq;
#pragma unroll
  for (int ai = 0; ai < 2; ++ai)
#pragma unroll
    for (int m = 0; m < 4; ++m) {
      unsigned short* rowp = Cout + (size_t)(pm * BM + ai * HALF + wr * 64 + m * 16 + fr) * N + pn * BM + wc * 32 + (fq & 1) * 16 + (fq >> 1) * 8;
#pragma unroll
      for (int bj = 0; bj < 2; ++bj) {
        f32x4 v0 = acc[ai][bj][m][0], v1 = acc[ai][bj][m][1];
        unsigned x0 = cvt_pk_bf16(fmaxf(v0[0], 0.f), fmaxf(v0[1], 0.f)), x1 = cvt_pk_bf16(fmaxf(v0[2], 0.f), fmaxf(v0[3], 0.f));
        unsigned y0 = cvt_pk_bf16(fmaxf(v1[0], 0.f), fmaxf(v1[1], 0.f)), y1 = cvt_pk_bf16(fmaxf(v1[2], 0.f), fmaxf(v1[3], 0.f));
        auto r0 = __builtin_amdgcn_permlane16_swap(x0, y0, false, false);
        auto r1 = __builtin_amdgcn_permlane16_swap(x1, y1, false, false);
        u32x4 w; w.x = r0[0]; w.y = r1[0]; w.z = r0[1]; w.w = r1[1];
        if (bad) w = (u32x4){0x7fc07fc0u, 0x7fc07fc0u, 0x7fc07fc0u, 0x7fc07fc0u};
        *(u32x4*)(rowp + bj * HALF) = w;
      }
    }
}

}

#define HIP_CHECK(expr) do { hipError_t _e = (expr); if (_e != hipSuccess) { \
    fprintf(stderr, "HIP error %s at %s:%d\n", hipGetErrorString(_e), __FILE__, __LINE__); \
    return; } } while (0)

extern "C" void kernel_launch(int rank, int nranks, ncclComm_t comm,
                              hipStream_t stream,
                              void* const* d_in, const int* in_sizes, int n_in,
                              void* d_out, int out_size,
                              const int* in_dtypes, int out_dtype,
                              void* const* d_out_all,
                              void* const* d_ws_all, size_t ws_bytes) {
  if (nranks != NRANK || ws_bytes < WS_NEED || n_in < 2 || in_sizes[0] != MSH * KDIM || in_sizes[1] != KDIM * NLOC || out_size != M_ALL * NLOC) {
    fprintf(stderr, "unexpected configuration\n"); return;
  }
  static bool init[64] = {};
  int dev = 0; HIP_CHECK(hipGetDevice(&dev)); dev &= 63;
  if (!init[dev]) {
    HIP_CHECK(hipFuncSetAttribute((const void*)ag_gemm_fused, hipFuncAttributeMaxDynamicSharedMemorySize, LDS_BYTES));
    init[dev] = true;
  }
  ag_gemm_fused<<<NWG, 512, LDS_BYTES, stream>>>((const unsigned short*)d_in[0], (const unsigned short*)d_in[1],
                                                 (unsigned short*)d_out, d_ws_all, rank);
  HIP_CHECK(hipGetLastError());
}
```

```cpp
#include <hip/hip_runtime.h>
#include <rccl/rccl.h>
#include <cstdio>

#define LAS __attribute__((address_space(3)))
typedef short bf16x8 __attribute__((ext_vector_type(8)));
typedef short bf16x4 __attribute__((ext_vector_type(4)));
typedef float f32x4 __attribute__((ext_vector_type(4)));
typedef unsigned u32x4 __attribute__((ext_vector_type(4)));
typedef unsigned long long u64;

namespace {

constexpr int M_ALL = 4096, KDIM = 4096, NLOC = 4096, NRANK = 8, MSH = M_ALL / NRANK;
constexpr int BM = 256, BK = 64, HALF = 128, HTB = HALF * BK * 2, NXCD = 8, WGM = 8;
constexpr int STAGE_BYTES = 8 * HTB;
constexpr int LDS_BYTES = STAGE_BYTES + 64;
constexpr int NWG = (M_ALL / BM) * (NLOC / BM);
constexpr int CHUNK_COLS = 256, NCHUNK = KDIM / CHUNK_COLS;
constexpr size_t WS_CTR = 0;
constexpr size_t WS_TMO = 2048;
constexpr size_t WS_FLAGS = 4096;
constexpr size_t WS_STAMP = 512u << 10;
constexpr size_t WS_GATHER = 1u << 20;
constexpr size_t GATHER_BYTES = (size_t)M_ALL * KDIM * 2;
constexpr size_t WS_NEED = WS_GATHER + 2 * GATHER_BYTES;
#define SLOTS 3
constexpr u64 DEADLINE_TICKS = 20000000ull;

__host__ __device__ __forceinline__ int lds_byte(int r, int c) {
  const int st = (r >> 4) * 2 + (c >> 5), rr = r & 15, cc = c & 31, ob = rr * 64 + cc * 2;
  return st * 1024 + (ob ^ (((ob >> 9) & 1) << 5));
}
__host__ __device__ __forceinline__ void stage_rc(int b, int& R, int& C) {
  const int st = b / 1024, sb = b % 1024, swz = sb ^ (((sb >> 9) & 1) << 5);
  R = (st >> 1) * 16 + swz / 64; C = (st & 1) * 32 + (swz % 64) / 2;
}
__host__ __device__ __forceinline__ int fswz(int row) { return ((row & 3) << 2) | ((row >> 2) & 3); }

__device__ __forceinline__ unsigned cvt_pk_bf16(float lo, float hi) {
  unsigned r; asm volatile("v_cvt_pk_bf16_f32 %0, %1, %2" : "=v"(r) : "v"(lo), "v"(hi)); return r;
}
template <int OFF> __device__ __forceinline__ bf16x8 tr_frag(unsigned a0, unsigned a1) {
  bf16x4 lo, hi;
  asm volatile("ds_read_b64_tr_b16 %0, %1 offset:%2" : "=&v"(lo) : "v"(a0), "i"(OFF));
  asm volatile("ds_read_b64_tr_b16 %0, %1 offset:%2" : "=&v"(hi) : "v"(a1), "i"(OFF));
  bf16x8 r;
  r[0] = lo[0]; r[1] = lo[1]; r[2] = lo[2]; r[3] = lo[3];
  r[4] = hi[0]; r[5] = hi[1]; r[6] = hi[2]; r[7] = hi[3];
  return r;
}
__device__ __forceinline__ void store16_sys(u64 base, unsigned voff, u32x4 v) {
#define STORE_MOD "sc0 sc1"
  asm volatile("global_store_dwordx4 %0, %1, %2 " STORE_MOD "\n\ts_nop 1" :: "v"(voff), "v"(v), "s"(base) : "memory");
}
__device__ __forceinline__ u64 lane_ptr(unsigned lo, unsigned hi, int p) {
  unsigned l = __builtin_amdgcn_readlane(lo, p), h = __builtin_amdgcn_readlane(hi, p);
  return ((u64)h << 32) | l;
}

__device__ __forceinline__ bool wait_flags(const u64* fp, int lane, unsigned seq, unsigned* tmo) {
  const u64 t0 = __builtin_amdgcn_s_memrealtime();
  for (;;) {
    u64 v = __hip_atomic_load(fp + lane, __ATOMIC_RELAXED, __HIP_MEMORY_SCOPE_SYSTEM);
    bool ok = ((unsigned)v - seq) < 0x80000000u && ((unsigned)(v >> 32) - seq) < 0x80000000u;
    if (__all(ok)) return true;
    unsigned tm = __hip_atomic_load(tmo, __ATOMIC_RELAXED, __HIP_MEMORY_SCOPE_AGENT);
    if (__builtin_amdgcn_readfirstlane(tm) == seq || __builtin_amdgcn_s_memrealtime() - t0 > DEADLINE_TICKS) {
      if (lane == 0) __hip_atomic_store(tmo, seq, __ATOMIC_RELAXED, __HIP_MEMORY_SCOPE_AGENT);
      return false;
    }
    __builtin_amdgcn_s_sleep(1);
  }
}

__global__ __launch_bounds__(512, 2)
void ag_gemm_fused(const unsigned short* __restrict__ X, const unsigned short* __restrict__ W, unsigned short* __restrict__ Cout,
                   void* const* __restrict__ ws_all, int rank) {
  extern __shared__ __attribute__((aligned(16))) unsigned char smem[];
  LAS unsigned char* lds = (LAS unsigned char*)smem;
  const int tid = threadIdx.x, wid = __builtin_amdgcn_readfirstlane(tid >> 6), lane = tid & 63, wr = wid >> 2, wc = wid & 3, fr = lane & 15, fq = lane >> 4;
  constexpr int K = KDIM, N = NLOC, nt = K / BK;
  unsigned char* myws = (unsigned char*)ws_all[rank];
#define STAMP() do {} while (0)
  STAMP();

  if (tid == 0) {
    unsigned old = __hip_atomic_fetch_add((unsigned*)(myws + WS_CTR + (blockIdx.x & 7) * 128), 1u, __ATOMIC_RELAXED, __HIP_MEMORY_SCOPE_AGENT);
    *(LAS unsigned*)(lds + STAGE_BYTES) = old / (NWG / 8) + 1;
  }
  u64 pw = (u64)ws_all[lane & 7];
  const unsigned xoff = (unsigned)(((2 * blockIdx.x + (lane >> 5)) * K + (lane & 31) * 8) * 2);
  u32x4 xd0 = *(const u32x4*)((const char*)X + xoff + wid * 512);
  u32x4 xd1 = *(const u32x4*)((const char*)X + xoff + (wid + 8) * 512);
  asm volatile("" : "+v"(pw), "+v"(xd0), "+v"(xd1));
  const unsigned pw_lo = (unsigned)pw, pw_hi = (unsigned)(pw >> 32);
  __syncthreads();
  const unsigned seq = __builtin_amdgcn_readfirstlane(*(LAS unsigned*)(lds + STAGE_BYTES));
  STAMP();
  const size_t gbuf = WS_GATHER + (size_t)(seq & 1) * GATHER_BYTES;
  unsigned* tmo = (unsigned*)(myws + WS_TMO);

  const unsigned doff = (unsigned)(rank * MSH * K * 2) + xoff;
  const int rot = blockIdx.x % 7;
  int ns = 0;
#define WAIT_VS(k) WAIT_VS_(k)
#define WAIT_VS_(k) do { if ((k) == 0) WAIT_V(6); else if ((k) == 1) WAIT_V(7); else if ((k) == 2) WAIT_V(8); else if ((k) == 3) WAIT_V(9); else WAIT_V(10); } while (0)
#define DBG_PEER(x) (x)
#define PUSH_Q(c, q) do { int q_ = (q) + rot; q_ = q_ >= 7 ? q_ - 7 : q_; \
      u64 b_ = lane_ptr(pw_lo, pw_hi, DBG_PEER((rank + 1 + q_) & 7)) + gbuf; \
      store16_sys(b_, doff + (c) * 512, (c) < 8 ? xd0 : xd1); } while (0)
#define SEND_FLAG(c) do { if (lane < 8 && lane != rank) { \
      unsigned* f_ = (unsigned*)(pw + WS_FLAGS) + (rank * NCHUNK + (c)) * NWG + blockIdx.x; \
      asm volatile("global_store_dword %0, %1, off sc0 sc1\n\ts_nop 1" :: "v"(f_), "v"(seq) : "memory"); } } while (0)
#define LA 2
  if (wid < LA) {
#define STAGGER 350
    if (wid >= 1) { const u64 t0_ = __builtin_amdgcn_s_memrealtime(); while (__builtin_amdgcn_s_memrealtime() - t0_ < STAGGER * wid) __builtin_amdgcn_s_sleep(8); }
#pragma unroll
    for (int q = 0; q < 7; ++q) PUSH_Q(wid, q);
    if (wid == 0) { asm volatile("s_waitcnt vmcnt(0)" ::: "memory"); SEND_FLAG(0); }
  }

  int wgid = blockIdx.x;
  { const int xcd = wgid % NXCD, off = wgid / NXCD; wgid = xcd * (NWG / NXCD) + off; }
  constexpr int nN = N / BM, nig = WGM * nN;
  const int pm = (wgid / nig) * WGM + (wgid % nig) % WGM, pn = (wgid % nig) / WGM;
  const int src = pm >> 1;
  const bool remote = (src != rank);

#define SA(b, h) (((b) * 2 + (h)) * HTB)
#define SB(b, h) ((4 + (b) * 2 + (h)) * HTB)
  unsigned voffA[2], voffB[2];
#pragma unroll
  for (int i = 0; i < 2; ++i) {
    int R, C; stage_rc(tid * 16 + i * 8192, R, C);
    voffA[i] = (unsigned)(R * K + C) * 2u;
    int b = tid * 16 + i * 8192, row = b >> 8, chp = (b >> 4) & 15, ch = chp ^ fswz(row);
    voffB[i] = (unsigned)(row * N + ch * 8) * 2u;
  }
  constexpr size_t kstepA = (size_t)(BK * 2), hstepA = (size_t)HALF * K * 2, kstepB = (size_t)BK * N * 2, hstepB = (size_t)HALF * 2;
  const unsigned ldsw = (unsigned)wid * 1024u;
  const int aoff = lds_byte(wr * 64 + fr, fq * 8);
  int boff[2][2];
  { const int g = fq, q = fr >> 2, p = lane & 3;
#pragma unroll
    for (int n = 0; n < 2; ++n)
#pragma unroll
      for (int t = 0; t < 2; ++t) {
        int row = 8 * g + 4 * t + q; int chunk = 2 * (2 * wc + n) + (p >> 1);
        boff[n][t] = SB(0, 0) + 256 * row + 16 * (chunk ^ fswz(row)) + 8 * (p & 1);
      }
  }
#define STAGE(bufoff, gbase, voff) do { _Pragma("unroll") for (int _i = 0; _i < 2; ++_i) \
    __builtin_amdgcn_global_load_lds((const unsigned*)((const char*)(gbase) + (voff)[_i]), (LAS unsigned*)(lds + (bufoff) + ldsw + _i * 8192), 16, 0, 0); } while (0)
#define LDA(dst, b, h) do { _Pragma("unroll") for (int m = 0; m < 4; ++m) _Pragma("unroll") for (int k = 0; k < 2; ++k) dst[m][k] = *(const LAS bf16x8*)(lds + SA(b, h) + aoff + m * 2048 + k * 1024); } while (0)
#define LDB(dst, b, h) do { _Pragma("unroll") for (int n = 0; n < 2; ++n) { dst[n][0] = tr_frag<SB(b, h) - SB(0, 0)>(boff[n][0], boff[n][1]); dst[n][1] = tr_frag<SB(b, h) - SB(0, 0) + 8192>(boff[n][0], boff[n][1]); } } while (0)
#define MMA(ai, bj, At, Bt) do { __builtin_amdgcn_s_setprio(1); _Pragma("unroll") for (int m = 0; m < 4; ++m) _Pragma("unroll") for (int n = 0; n < 2; ++n) _Pragma("unroll") for (int k = 0; k < 2; ++k) \
    acc[ai][bj][m][n] = __builtin_amdgcn_mfma_f32_16x16x32_bf16(Bt[n][k], At[m][k], acc[ai][bj][m][n], 0, 0, 0); __builtin_amdgcn_s_setprio(0); } while (0)
#define WAIT_V(n) asm volatile("s_waitcnt vmcnt(" #n ")" ::: "memory")
#define WAIT_L(n) asm volatile("s_waitcnt lgkmcnt(" #n ")" ::: "memory")
#define BAR __builtin_amdgcn_s_barrier()
#define SCHED __builtin_amdgcn_sched_barrier(0)

  f32x4 acc[2][2][4][2];
#pragma unroll
  for (int a = 0; a < 2; ++a)
#pragma unroll
    for (int b = 0; b < 2; ++b)
#pragma unroll
      for (int m = 0; m < 4; ++m)
#pragma unroll
        for (int n = 0; n < 2; ++n) acc[a][b][m][n] = (f32x4){0.f, 0.f, 0.f, 0.f};
  bf16x8 At[4][2], B0[2][2], B1[2][2];
#define GB_RD gbuf
  const char* cA = (remote ? (const char*)(myws + GB_RD) + (size_t)pm * 2 * hstepA : (const char*)X + (size_t)(pm & 1) * 2 * hstepA);
  const char* cB = (const char*)W + (size_t)pn * 2 * hstepB;
  const u64* myflags = (const u64*)((unsigned*)(myws + WS_FLAGS) + (size_t)src * NCHUNK * NWG + (pm & 1) * 128);

  if (remote && wid == 0) { wait_flags(myflags, lane, seq, tmo); }
  asm volatile("" ::: "memory");
  __syncthreads();

  STAMP();
  STAGE(SB(0, 0), cB, voffB); STAGE(SA(0, 0), cA, voffA); STAGE(SB(0, 1), cB + hstepB, voffB); STAGE(SA(0, 1), cA + hstepA, voffA);
  if (wr == 1) BAR;
  WAIT_V(4); BAR;
  STAGE(SB(1, 0), cB + kstepB, voffB); STAGE(SA(1, 0), cA + kstepA, voffA); STAGE(SB(1, 1), cB + hstepB + kstepB, voffB);
  WAIT_V(6); BAR;
  constexpr int FLAG_H = 6 / SLOTS + 1;
#define SLOTS_HALF(h) do { ns = 0; if (act) { _Pragma("unroll") for (int s_ = 0; s_ < SLOTS; ++s_) { const int q = (h) * SLOTS + s_; if (q < 7) { PUSH_Q(ca, q); ++ns; } } } } while (0)
#define FLAG_HALF(h) do { if (FLAG_H < 4) { if (act && (h) == FLAG_H) SEND_FLAG(ca); if (t == 0 && (h) == 0 && wid >= 1 && wid < LA) SEND_FLAG(wid); } \
      else if ((h) == 0) { if (t == 0) { if (wid >= 1 && wid < LA) SEND_FLAG(wid); } else if (ca - 1 < NCHUNK && wid == ((ca - 1) & 7)) SEND_FLAG(ca - 1); } } while (0)
  u64 pv = 0;
  for (int t = 0; t < nt; t += 2) {
    const bool last = (t == nt - 2);
    STAMP();
    const char* a1 = cA + (size_t)(t + 1) * kstepA;
    const char* a2 = last ? cA : cA + (size_t)(t + 2) * kstepA; const char* b2 = last ? cB : cB + (size_t)(t + 2) * kstepB;
    const char* a3 = a2 + kstepA; const char* b3 = b2 + kstepB;
    const int ca = (t >> 2) + LA;
    const bool act = ca < NCHUNK && wid == (ca & 7);
    const int h0 = t & 2;
    if (remote && wid == 0 && (t & 2) && !last) {
      const bool ok = ((unsigned)pv - seq) < 0x80000000u && ((unsigned)(pv >> 32) - seq) < 0x80000000u;
      if (!__all(ok)) wait_flags(myflags + (size_t)((t + 2) >> 2) * (NWG / 2), lane, seq, tmo);
    }
    LDB(B0, 0, 0); SCHED; LDA(At, 0, 0); STAGE(SA(1, 1), a1 + hstepA, voffA); SLOTS_HALF(h0);
    WAIT_L(8); BAR; WAIT_L(0); MMA(0, 0, At, B0); BAR; SCHED;
    LDB(B1, 0, 1); STAGE(SB(0, 0), b2, voffB);
    BAR; WAIT_L(0); MMA(0, 1, At, B1); BAR;
    LDA(At, 0, 1); STAGE(SA(0, 0), a2, voffA);
    BAR; WAIT_L(0); MMA(1, 0, At, B0); BAR; SCHED;
    STAGE(SB(0, 1), b2 + hstepB, voffB);
    WAIT_VS(ns); FLAG_HALF(h0); BAR; MMA(1, 1, At, B1); BAR;
    if (remote && wid == 0 && !(t & 2)) asm volatile("global_load_dwordx2 %0, %1, off sc0 sc1" : "=v"(pv) : "v"(myflags + (size_t)((t >> 2) + 1) * (NWG / 2) + lane) : "memory");
    LDB(B0, 1, 0); SCHED; LDA(At, 1, 0); STAGE(SA(0, 1), a2 + hstepA, voffA); SLOTS_HALF(h0 + 1);
    WAIT_L(8); BAR; WAIT_L(0); MMA(0, 0, At, B0); BAR; SCHED;
    LDB(B1, 1, 1); STAGE(SB(1, 0), b3, voffB);
    BAR; WAIT_L(0); MMA(0, 1, At, B1); BAR;
    LDA(At, 1, 1); STAGE(SA(1, 0), a3, voffA);
    BAR; WAIT_L(0); MMA(1, 0, At, B0); BAR; SCHED;
    STAGE(SB(1, 1), b3 + hstepB, voffB);
    WAIT_VS(ns); FLAG_HALF(h0 + 1);
    BAR; MMA(1, 1, At, B1); BAR;
  }
  WAIT_V(0);
  if (wr == 0) BAR;
  STAMP();
  const bool bad = __builtin_amdgcn_readfirstlane(__hip_atomic_load(tmo, __ATOMIC_RELAXED, __HIP_MEMORY_SCOPE_AGENT)) == seq;
#pragma unroll
  for (int ai = 0; ai < 2; ++ai)
#pragma unroll
    for (int m = 0; m < 4; ++m) {
      unsigned short* rowp = Cout + (size_t)(pm * BM + ai * HALF + wr * 64 + m * 16 + fr) * N + pn * BM + wc * 32 + (fq & 1) * 16 + (fq >> 1) * 8;
#pragma unroll
      for (int bj = 0; bj < 2; ++bj) {
        f32x4 v0 = acc[ai][bj][m][0], v1 = acc[ai][bj][m][1];
        unsigned x0 = cvt_pk_bf16(fmaxf(v0[0], 0.f), fmaxf(v0[1], 0.f)), x1 = cvt_pk_bf16(fmaxf(v0[2], 0.f), fmaxf(v0[3], 0.f));
        unsigned y0 = cvt_pk_bf16(fmaxf(v1[0], 0.f), fmaxf(v1[1], 0.f)), y1 = cvt_pk_bf16(fmaxf(v1[2], 0.f), fmaxf(v1[3], 0.f));
        auto r0 = __builtin_amdgcn_permlane16_swap(x0, y0, false, false);
        auto r1 = __builtin_amdgcn_permlane16_swap(x1, y1, false, false);
        u32x4 w; w.x = r0[0]; w.y = r1[0]; w.z = r0[1]; w.w = r1[1];
        if (bad) w = (u32x4){0x7fc07fc0u, 0x7fc07fc0u, 0x7fc07fc0u, 0x7fc07fc0u};
        *(u32x4*)(rowp + bj * HALF) = w;
      }
    }
}

}

#define HIP_CHECK(expr) do { hipError_t _e = (expr); if (_e != hipSuccess) { \
    fprintf(stderr, "HIP error %s at %s:%d\n", hipGetErrorString(_e), __FILE__, __LINE__); \
    return; } } while (0)

extern "C" void kernel_launch(int rank, int nranks, ncclComm_t comm,
                              hipStream_t stream,
                              void* const* d_in, const int* in_sizes, int n_in,
                              void* d_out, int out_size,
                              const int* in_dtypes, int out_dtype,
                              void* const* d_out_all,
                              void* const* d_ws_all, size_t ws_bytes) {
  if (nranks != NRANK || ws_bytes < WS_NEED || n_in < 2 || in_sizes[0] != MSH * KDIM || in_sizes[1] != KDIM * NLOC || out_size != M_ALL * NLOC) {
    fprintf(stderr, "unexpected configuration\n"); return;
  }
  static bool init[64] = {};
  int dev = 0; HIP_CHECK(hipGetDevice(&dev)); dev &= 63;
  if (!init[dev]) {
    HIP_CHECK(hipFuncSetAttribute((const void*)ag_gemm_fused, hipFuncAttributeMaxDynamicSharedMemorySize, LDS_BYTES));
    init[dev] = true;
  }
  ag_gemm_fused<<<NWG, 512, LDS_BYTES, stream>>>((const unsigned short*)d_in[0], (const unsigned short*)d_in[1],
                                                 (unsigned short*)d_out, d_ws_all, rank);
  HIP_CHECK(hipGetLastError());
}
```

```cpp
#include <hip/hip_runtime.h>
#include <rccl/rccl.h>
#include <cstdio>

#define LAS __attribute__((address_space(3)))
typedef short bf16x8 __attribute__((ext_vector_type(8)));
typedef short bf16x4 __attribute__((ext_vector_type(4)));
typedef float f32x4 __attribute__((ext_vector_type(4)));
typedef unsigned u32x4 __attribute__((ext_vector_type(4)));
typedef unsigned long long u64;

namespace {

constexpr int M_ALL = 4096, KDIM = 4096, NLOC = 4096, NRANK = 8, MSH = M_ALL / NRANK;
constexpr int BM = 256, BK = 64, HALF = 128, HTB = HALF * BK * 2, NXCD = 8, WGM = 8;
constexpr int STAGE_BYTES = 8 * HTB;
constexpr int LDS_BYTES = STAGE_BYTES + 64;
constexpr int NWG = (M_ALL / BM) * (NLOC / BM);
constexpr int CHUNK_COLS = 256, NCHUNK = KDIM / CHUNK_COLS;
constexpr size_t WS_CTR = 0;
constexpr size_t WS_TMO = 2048;
constexpr size_t WS_FLAGS = 4096;
constexpr size_t WS_GATHER = 1u << 20;
constexpr size_t GATHER_BYTES = (size_t)M_ALL * KDIM * 2;
constexpr size_t WS_NEED = WS_GATHER + 2 * GATHER_BYTES;
constexpr int SLOTS = 2;
constexpr int LA = 2;
constexpr unsigned long long STAGGER = 350;
constexpr u64 DEADLINE_TICKS = 20000000ull;

__host__ __device__ __forceinline__ int lds_byte(int r, int c) {
  const int st = (r >> 4) * 2 + (c >> 5), rr = r & 15, cc = c & 31, ob = rr * 64 + cc * 2;
  return st * 1024 + (ob ^ (((ob >> 9) & 1) << 5));
}
__host__ __device__ __forceinline__ void stage_rc(int b, int& R, int& C) {
  const int st = b / 1024, sb = b % 1024, swz = sb ^ (((sb >> 9) & 1) << 5);
  R = (st >> 1) * 16 + swz / 64; C = (st & 1) * 32 + (swz % 64) / 2;
}
__host__ __device__ __forceinline__ int fswz(int row) { return ((row & 3) << 2) | ((row >> 2) & 3); }

__device__ __forceinline__ unsigned cvt_pk_bf16(float lo, float hi) {
  unsigned r; asm volatile("v_cvt_pk_bf16_f32 %0, %1, %2" : "=v"(r) : "v"(lo), "v"(hi)); return r;
}
template <int OFF> __device__ __forceinline__ bf16x8 tr_frag(unsigned a0, unsigned a1) {
  bf16x4 lo, hi;
  asm volatile("ds_read_b64_tr_b16 %0, %1 offset:%2" : "=&v"(lo) : "v"(a0), "i"(OFF));
  asm volatile("ds_read_b64_tr_b16 %0, %1 offset:%2" : "=&v"(hi) : "v"(a1), "i"(OFF));
  bf16x8 r;
  r[0] = lo[0]; r[1] = lo[1]; r[2] = lo[2]; r[3] = lo[3];
  r[4] = hi[0]; r[5] = hi[1]; r[6] = hi[2]; r[7] = hi[3];
  return r;
}
__device__ __forceinline__ void store16_sys(u64 base, unsigned voff, u32x4 v) {
  asm volatile("global_store_dwordx4 %0, %1, %2  sc0 sc1\n\ts_nop 1" :: "v"(voff), "v"(v), "s"(base) : "memory");
}
__device__ __forceinline__ u64 lane_ptr(unsigned lo, unsigned hi, int p) {
  unsigned l = __builtin_amdgcn_readlane(lo, p), h = __builtin_amdgcn_readlane(hi, p);
  return ((u64)h << 32) | l;
}

__device__ __forceinline__ bool wait_flags(const u64* fp, int lane, unsigned seq, unsigned* tmo) {
  const u64 t0 = __builtin_amdgcn_s_memrealtime();
  for (;;) {
    u64 v = __hip_atomic_load(fp + lane, __ATOMIC_RELAXED, __HIP_MEMORY_SCOPE_SYSTEM);
    bool ok = ((unsigned)v - seq) < 0x80000000u && ((unsigned)(v >> 32) - seq) < 0x80000000u;
    if (__all(ok)) return true;
    unsigned tm = __hip_atomic_load(tmo, __ATOMIC_RELAXED, __HIP_MEMORY_SCOPE_AGENT);
    if (__builtin_amdgcn_readfirstlane(tm) == seq || __builtin_amdgcn_s_memrealtime() - t0 > DEADLINE_TICKS) {
      if (lane == 0) __hip_atomic_store(tmo, seq, __ATOMIC_RELAXED, __HIP_MEMORY_SCOPE_AGENT);
      return false;
    }
    __builtin_amdgcn_s_sleep(1);
  }
}

__global__ __launch_bounds__(512, 2)
void ag_gemm_fused(const unsigned short* __restrict__ X, const unsigned short* __restrict__ W, unsigned short* __restrict__ Cout,
                   void* const* __restrict__ ws_all, int rank) {
  extern __shared__ __attribute__((aligned(16))) unsigned char smem[];
  LAS unsigned char* lds = (LAS unsigned char*)smem;
  const int tid = threadIdx.x, wid = __builtin_amdgcn_readfirstlane(tid >> 6), lane = tid & 63, wr = wid >> 2, wc = wid & 3, fr = lane & 15, fq = lane >> 4;
  constexpr int K = KDIM, N = NLOC, nt = K / BK;
  unsigned char* myws = (unsigned char*)ws_all[rank];

  if (tid == 0) {
    unsigned old = __hip_atomic_fetch_add((unsigned*)(myws + WS_CTR + (blockIdx.x & 7) * 128), 1u, __ATOMIC_RELAXED, __HIP_MEMORY_SCOPE_AGENT);
    *(LAS unsigned*)(lds + STAGE_BYTES) = old / (NWG / 8) + 1;
  }
  u64 pw = (u64)ws_all[lane & 7];
  const unsigned xoff = (unsigned)(((2 * blockIdx.x + (lane >> 5)) * K + (lane & 31) * 8) * 2);
  u32x4 xd0 = *(const u32x4*)((const char*)X + xoff + wid * 512);
  u32x4 xd1 = *(const u32x4*)((const char*)X + xoff + (wid + 8) * 512);
  asm volatile("" : "+v"(pw), "+v"(xd0), "+v"(xd1));
  const unsigned pw_lo = (unsigned)pw, pw_hi = (unsigned)(pw >> 32);
  __syncthreads();
  const unsigned seq = __builtin_amdgcn_readfirstlane(*(LAS unsigned*)(lds + STAGE_BYTES));
  const size_t gbuf = WS_GATHER + (size_t)(seq & 1) * GATHER_BYTES;
  unsigned* tmo = (unsigned*)(myws + WS_TMO);

  const unsigned doff = (unsigned)(rank * MSH * K * 2) + xoff;
  const int rot = blockIdx.x % 7;
  int ns = 0;
#define WAIT_VS(k) WAIT_VS_(k)
#define WAIT_VS_(k) do { if ((k) == 0) WAIT_V(6); else if ((k) == 1) WAIT_V(7); else if ((k) == 2) WAIT_V(8); else if ((k) == 3) WAIT_V(9); else WAIT_V(10); } while (0)
#define PUSH_Q(c, q) do { int q_ = (q) + rot; q_ = q_ >= 7 ? q_ - 7 : q_; \
      u64 b_ = lane_ptr(pw_lo, pw_hi, (rank + 1 + q_) & 7) + gbuf; \
      store16_sys(b_, doff + (c) * 512, (c) < 8 ? xd0 : xd1); } while (0)
#define SEND_FLAG(c) do { if (lane < 8 && lane != rank) { \
      unsigned* f_ = (unsigned*)(pw + WS_FLAGS) + (rank * NCHUNK + (c)) * NWG + blockIdx.x; \
      asm volatile("global_store_dword %0, %1, off sc0 sc1\n\ts_nop 1" :: "v"(f_), "v"(seq) : "memory"); } } while (0)
  if (wid < LA) {
    if (wid >= 1) { const u64 t0_ = __builtin_amdgcn_s_memrealtime(); while (__builtin_amdgcn_s_memrealtime() - t0_ < STAGGER * wid) __builtin_amdgcn_s_sleep(8); }
#pragma unroll
    for (int q = 0; q < 7; ++q) PUSH_Q(wid, q);
    if (wid == 0) {
#pragma unroll
      for (int q = 0; q < 7; ++q) {
        int q_ = q + rot; q_ = q_ >= 7 ? q_ - 7 : q_;
        asm volatile("s_waitcnt vmcnt(6)" ::: "memory");
        if (lane == ((rank + 1 + q_) & 7)) {
          unsigned* f_ = (unsigned*)(pw + WS_FLAGS) + (rank * NCHUNK) * NWG + blockIdx.x;
          asm volatile("global_store_dword %0, %1, off sc0 sc1\n\ts_nop 1" :: "v"(f_), "v"(seq) : "memory");
        }
      }
    }
  }

  int wgid = blockIdx.x;
  { const int xcd = wgid % NXCD, off = wgid / NXCD; wgid = xcd * (NWG / NXCD) + off; }
  constexpr int nN = N / BM, nig = WGM * nN;
  const int pm = (wgid / nig) * WGM + (wgid % nig) % WGM, pn = (wgid % nig) / WGM;
  const int src = pm >> 1;
  const bool remote = (src != rank);

#define SA(b, h) (((b) * 2 + (h)) * HTB)
#define SB(b, h) ((4 + (b) * 2 + (h)) * HTB)
  unsigned voffA[2], voffB[2];
#pragma unroll
  for (int i = 0; i < 2; ++i) {
    int R, C; stage_rc(tid * 16 + i * 8192, R, C);
    voffA[i] = (unsigned)(R * K + C) * 2u;
    int b = tid * 16 + i * 8192, row = b >> 8, chp = (b >> 4) & 15, ch = chp ^ fswz(row);
    voffB[i] = (unsigned)(row * N + ch * 8) * 2u;
  }
  constexpr size_t kstepA = (size_t)(BK * 2), hstepA = (size_t)HALF * K * 2, kstepB = (size_t)BK * N * 2, hstepB = (size_t)HALF * 2;
  const unsigned ldsw = (unsigned)wid * 1024u;
  const int aoff = lds_byte(wr * 64 + fr, fq * 8);
  int boff[2][2];
  { const int g = fq, q = fr >> 2, p = lane & 3;
#pragma unroll
    for (int n = 0; n < 2; ++n)
#pragma unroll
      for (int t = 0; t < 2; ++t) {
        int row = 8 * g + 4 * t + q; int chunk = 2 * (2 * wc + n) + (p >> 1);
        boff[n][t] = SB(0, 0) + 256 * row + 16 * (chunk ^ fswz(row)) + 8 * (p & 1);
      }
  }
#define STAGE(bufoff, gbase, voff) do { _Pragma("unroll") for (int _i = 0; _i < 2; ++_i) \
    __builtin_amdgcn_global_load_lds((const unsigned*)((const char*)(gbase) + (voff)[_i]), (LAS unsigned*)(lds + (bufoff) + ldsw + _i * 8192), 16, 0, 0); } while (0)
#define LDA(dst, b, h) do { _Pragma("unroll") for (int m = 0; m < 4; ++m) _Pragma("unroll") for (int k = 0; k < 2; ++k) dst[m][k] = *(const LAS bf16x8*)(lds + SA(b, h) + aoff + m * 2048 + k * 1024); } while (0)
#define LDB(dst, b, h) do { _Pragma("unroll") for (int n = 0; n < 2; ++n) { dst[n][0] = tr_frag<SB(b, h) - SB(0, 0)>(boff[n][0], boff[n][1]); dst[n][1] = tr_frag<SB(b, h) - SB(0, 0) + 8192>(boff[n][0], boff[n][1]); } } while (0)
#define MMA(ai, bj, At, Bt) do { __builtin_amdgcn_s_setprio(1); _Pragma("unroll") for (int m = 0; m < 4; ++m) _Pragma("unroll") for (int n = 0; n < 2; ++n) _Pragma("unroll") for (int k = 0; k < 2; ++k) \
    acc[ai][bj][m][n] = __builtin_amdgcn_mfma_f32_16x16x32_bf16(Bt[n][k], At[m][k], acc[ai][bj][m][n], 0, 0, 0); __builtin_amdgcn_s_setprio(0); } while (0)
#define WAIT_V(n) asm volatile("s_waitcnt vmcnt(" #n ")" ::: "memory")
#define WAIT_L(n) asm volatile("s_waitcnt lgkmcnt(" #n ")" ::: "memory")
#define BAR __builtin_amdgcn_s_barrier()
#define SCHED __builtin_amdgcn_sched_barrier(0)

  f32x4 acc[2][2][4][2];
#pragma unroll
  for (int a = 0; a < 2; ++a)
#pragma unroll
    for (int b = 0; b < 2; ++b)
#pragma unroll
      for (int m = 0; m < 4; ++m)
#pragma unroll
        for (int n = 0; n < 2; ++n) acc[a][b][m][n] = (f32x4){0.f, 0.f, 0.f, 0.f};
  bf16x8 At[4][2], B0[2][2], B1[2][2];
  const char* cA = (remote ? (const char*)(myws + gbuf) + (size_t)pm * 2 * hstepA : (const char*)X + (size_t)(pm & 1) * 2 * hstepA);
  const char* cB = (const char*)W + (size_t)pn * 2 * hstepB;
  const u64* myflags = (const u64*)((unsigned*)(myws + WS_FLAGS) + (size_t)src * NCHUNK * NWG + (pm & 1) * 128);

  STAGE(SB(0, 0), cB, voffB); STAGE(SB(0, 1), cB + hstepB, voffB); STAGE(SB(1, 0), cB + kstepB, voffB); STAGE(SB(1, 1), cB + hstepB + kstepB, voffB);
  if (remote && wid == 0) { wait_flags(myflags, lane, seq, tmo); }
  asm volatile("" ::: "memory");
  __syncthreads();
  STAGE(SA(0, 0), cA, voffA); STAGE(SA(0, 1), cA + hstepA, voffA); STAGE(SA(1, 0), cA + kstepA, voffA);
  if (wr == 1) BAR;
  WAIT_V(2); BAR;
  BAR;
  constexpr int FLAG_H = 6 / SLOTS + 1;
#define SLOTS_HALF(h) do { ns = 0; if (act) { _Pragma("unroll") for (int s_ = 0; s_ < SLOTS; ++s_) { const int q = (h) * SLOTS + s_; if (q < 7) { PUSH_Q(ca, q); ++ns; } } } } while (0)
#define FLAG_HALF(h) do { if (FLAG_H < 4) { if (act && (h) == FLAG_H) SEND_FLAG(ca); if (t == 0 && (h) == 0 && wid >= 1 && wid < LA) SEND_FLAG(wid); } \
      else if ((h) == 0) { if (t == 0) { if (wid >= 1 && wid < LA) SEND_FLAG(wid); } else if (ca - 1 < NCHUNK && wid == ((ca - 1) & 7)) SEND_FLAG(ca - 1); } } while (0)
  u64 pv = 0;
  for (int t = 0; t < nt; t += 2) {
    const bool last = (t == nt - 2);
    const char* a1 = cA + (size_t)(t + 1) * kstepA;
    const char* a2 = last ? cA : cA + (size_t)(t + 2) * kstepA; const char* b2 = last ? cB : cB + (size_t)(t + 2) * kstepB;
    const char* a3 = a2 + kstepA; const char* b3 = b2 + kstepB;
    const int ca = (t >> 2) + LA;
    const bool act = ca < NCHUNK && wid == (ca & 7);
    const int h0 = t & 2;
    if (remote && wid == 0 && (t & 2) && !last) {
      const bool ok = ((unsigned)pv - seq) < 0x80000000u && ((unsigned)(pv >> 32) - seq) < 0x80000000u;
      if (!__all(ok)) wait_flags(myflags + (size_t)((t + 2) >> 2) * (NWG / 2), lane, seq, tmo);
    }
    LDB(B0, 0, 0); SCHED; LDA(At, 0, 0); STAGE(SA(1, 1), a1 + hstepA, voffA); SLOTS_HALF(h0);
    WAIT_L(8); BAR; WAIT_L(0); MMA(0, 0, At, B0); BAR; SCHED;
    LDB(B1, 0, 1); STAGE(SB(0, 0), b2, voffB);
    BAR; WAIT_L(0); MMA(0, 1, At, B1); BAR;
    LDA(At, 0, 1); STAGE(SA(0, 0), a2, voffA);
    BAR; WAIT_L(0); MMA(1, 0, At, B0); BAR; SCHED;
    STAGE(SB(0, 1), b2 + hstepB, voffB);
    WAIT_VS(ns); FLAG_HALF(h0); BAR; MMA(1, 1, At, B1); BAR;
    if (remote && wid == 0 && !(t & 2)) asm volatile("global_load_dwordx2 %0, %1, off sc0 sc1" : "=v"(pv) : "v"(myflags + (size_t)((t >> 2) + 1) * (NWG / 2) + lane) : "memory");
    LDB(B0, 1, 0); SCHED; LDA(At, 1, 0); STAGE(SA(0, 1), a2 + hstepA, voffA); SLOTS_HALF(h0 + 1);
    WAIT_L(8); BAR; WAIT_L(0); MMA(0, 0, At, B0); BAR; SCHED;
    LDB(B1, 1, 1); STAGE(SB(1, 0), b3, voffB);
    BAR; WAIT_L(0); MMA(0, 1, At, B1); BAR;
    LDA(At, 1, 1); STAGE(SA(1, 0), a3, voffA);
    BAR; WAIT_L(0); MMA(1, 0, At, B0); BAR; SCHED;
    STAGE(SB(1, 1), b3 + hstepB, voffB);
    WAIT_VS(ns); FLAG_HALF(h0 + 1);
    BAR; MMA(1, 1, At, B1); BAR;
  }
  WAIT_V(0);
  if (wr == 0) BAR;
  const bool bad = __builtin_amdgcn_readfirstlane(__hip_atomic_load(tmo, __ATOMIC_RELAXED, __HIP_MEMORY_SCOPE_AGENT)) == seq;
#pragma unroll
  for (int ai = 0; ai < 2; ++ai)
#pragma unroll
    for (int m = 0; m < 4; ++m) {
      unsigned short* rowp = Cout + (size_t)(pm * BM + ai * HALF + wr * 64 + m * 16 + fr) * N + pn * BM + wc * 32 + (fq & 1) * 16 + (fq >> 1) * 8;
#pragma unroll
      for (int bj = 0; bj < 2; ++bj) {
        f32x4 v0 = acc[ai][bj][m][0], v1 = acc[ai][bj][m][1];
        unsigned x0 = cvt_pk_bf16(fmaxf(v0[0], 0.f), fmaxf(v0[1], 0.f)), x1 = cvt_pk_bf16(fmaxf(v0[2], 0.f), fmaxf(v0[3], 0.f));
        unsigned y0 = cvt_pk_bf16(fmaxf(v1[0], 0.f), fmaxf(v1[1], 0.f)), y1 = cvt_pk_bf16(fmaxf(v1[2], 0.f), fmaxf(v1[3], 0.f));
        auto r0 = __builtin_amdgcn_permlane16_swap(x0, y0, false, false);
        auto r1 = __builtin_amdgcn_permlane16_swap(x1, y1, false, false);
        u32x4 w; w.x = r0[0]; w.y = r1[0]; w.z = r0[1]; w.w = r1[1];
        if (bad) w = (u32x4){0x7fc07fc0u, 0x7fc07fc0u, 0x7fc07fc0u, 0x7fc07fc0u};
        *(u32x4*)(rowp + bj * HALF) = w;
      }
    }
}

}

#define HIP_CHECK(expr) do { hipError_t _e = (expr); if (_e != hipSuccess) { \
    fprintf(stderr, "HIP error %s at %s:%d\n", hipGetErrorString(_e), __FILE__, __LINE__); \
    return; } } while (0)

extern "C" void kernel_launch(int rank, int nranks, ncclComm_t comm,
                              hipStream_t stream,
                              void* const* d_in, const int* in_sizes, int n_in,
                              void* d_out, int out_size,
                              const int* in_dtypes, int out_dtype,
                              void* const* d_out_all,
                              void* const* d_ws_all, size_t ws_bytes) {
  if (nranks != NRANK || ws_bytes < WS_NEED || n_in < 2 || in_sizes[0] != MSH * KDIM || in_sizes[1] != KDIM * NLOC || out_size != M_ALL * NLOC) {
    fprintf(stderr, "unexpected configuration\n"); return;
  }
  static bool init[64] = {};
  int dev = 0; HIP_CHECK(hipGetDevice(&dev)); dev &= 63;
  if (!init[dev]) {
    HIP_CHECK(hipFuncSetAttribute((const void*)ag_gemm_fused, hipFuncAttributeMaxDynamicSharedMemorySize, LDS_BYTES));
    init[dev] = true;
  }
  ag_gemm_fused<<<NWG, 512, LDS_BYTES, stream>>>((const unsigned short*)d_in[0], (const unsigned short*)d_in[1],
                                                 (unsigned short*)d_out, d_ws_all, rank);
  HIP_CHECK(hipGetLastError());
}
```
